```python
import numpy as np

kernel_name = "gpu_a2a_gemm_m2048_k2048_n2048_bf16_gelu"


ABSMAX_RTOL = 0.008

COMPUTE_DTYPE = "bfloat16"
EPILOGUE = "gelu"
IN_QUANT = None
OUT_QUANT = None
IN_DTYPES = {"x": "bfloat16", "w_mat": "bfloat16"}
WORLD_SIZE = 8

def _to_bf16(x):
    u = np.ascontiguousarray(x, dtype=np.float32).view(np.uint32)
    return ((u + 0x7FFF + ((u >> 16) & 1)) & 0xFFFF0000).view(np.float32)

def setup_inputs():

    x = _to_bf16(np.random.randn(2048, 2048).astype(np.float32))
    w_mat = _to_bf16((np.random.randn(2048, 2048) / np.sqrt(2048)).astype(np.float32))
    return {"x": x, "w_mat": w_mat}

def _epilogue(y):
    c = 0.7978845608028654
    return (0.5 * y * (1.0 + np.tanh(c * (y + 0.044715 * y ** 3)))).astype(np.float32)

def reference(x, w_mat):
    y = (x @ w_mat).astype(np.float32)
    return _epilogue(y)
```

```cpp
#include <hip/hip_runtime.h>
#include <rccl/rccl.h>
#include <cstdio>
#include <cstdint>

#define HIP_CHECK(expr) do { hipError_t _e = (expr); if (_e != hipSuccess) { \
    fprintf(stderr, "HIP error %s at %s:%d\n", hipGetErrorString(_e), __FILE__, __LINE__); \
    return; } } while (0)

typedef __attribute__((ext_vector_type(8))) __bf16 bf16x8;
typedef __attribute__((ext_vector_type(2))) __bf16 bf16x2;
typedef __attribute__((ext_vector_type(4))) float f32x4;
typedef __attribute__((ext_vector_type(2))) float f32x2;
typedef __attribute__((ext_vector_type(4))) unsigned u32x4;
typedef __attribute__((ext_vector_type(2))) unsigned u32x2;

namespace {

constexpr int NR = 8;
constexpr int MLOC = 256;
constexpr int KLOC = 256;
constexpr int KTOT = 2048, NTOT = 2048;
constexpr unsigned X_BYTES = 2048u * KLOC * 2u;
constexpr unsigned W_BYTES = (unsigned)KTOT * NTOT * 2u;
constexpr unsigned OUT_BYTES = MLOC * NTOT * 2u;
constexpr unsigned BUF_BYTES = 1u << 20;
constexpr unsigned SRC_BYTES = 128u << 10;
constexpr unsigned CNT_OFF = 3u * BUF_BYTES;
constexpr unsigned TMO_OFF = CNT_OFF + 256u * 128u;
constexpr unsigned PROF_OFF = 4u * BUF_BYTES;
constexpr unsigned WS_USED = 5u * BUF_BYTES;
constexpr unsigned long long DEADLINE_TICKS = 100000000ull;
constexpr int AUX_SYS = 17;

struct Args {
    const void* x;
    const void* w;
    void* out;
    void* ws[NR];
    int rank;
};

__device__ inline __amdgpu_buffer_rsrc_t rsrc(const void* p, unsigned bytes) {
    return __builtin_amdgcn_make_buffer_rsrc(const_cast<void*>(p), (short)0, (int)bytes, 0x00020000);
}

__device__ inline float gelu(float y) {
    const float k0 = -2.0f * 0.7978845608028654f * 1.4426950408889634f;
    const float k1 = k0 * 0.044715f;
    float t = __builtin_fmaf(y * y, k1, k0);
    float e = __builtin_amdgcn_exp2f(y * t);
    return y * __builtin_amdgcn_rcpf(1.0f + e);
}

#define STAMP(i) do {} while (0)

__global__ __launch_bounds__(512, 2) void a2a_gemm_gelu(Args a) {
    __shared__ __attribute__((aligned(16))) float red[8 * 32 * 64 + 4];

    const int tid = threadIdx.x;
    const int lane = tid & 63;
    const int wave = __builtin_amdgcn_readfirstlane(tid >> 6);
    const int b = blockIdx.x;
    const unsigned rank = (unsigned)a.rank & 7u;
    const int mt = (b >> 3) & 7;
    const int nt = ((b & 7) << 2) | (b >> 6);
    const unsigned src = (rank + (unsigned)wave) & 7u;
    STAMP(0);

    const __amdgpu_buffer_rsrc_t rx = rsrc(a.x, X_BYTES);
    const __amdgpu_buffer_rsrc_t rw = rsrc(a.w, W_BYTES);
    const __amdgpu_buffer_rsrc_t rme = rsrc(a.ws[rank], WS_USED);

    const unsigned dsel = (unsigned)(b >> 3) & 7u;
    const unsigned dest = (rank + 1u + dsel) & 7u;
    const bool pusher = (wave < 4) && (dsel != 7u);
    const int frag = 4 * ((b & 7) + 8 * (b >> 6)) + (wave & 3);
    u32x4 xv = {0u, 0u, 0u, 0u};
    if (pusher) {
        const int voff = (lane & 15) * (KLOC * 2) + (lane >> 4) * 16;
        const int soff = ((int)dest * MLOC + 16 * (frag >> 3)) * (KLOC * 2) + (frag & 7) * 64;
        xv = __builtin_amdgcn_raw_buffer_load_b128(rx, voff, soff, 0);
    }
    const unsigned cnt = __builtin_amdgcn_readfirstlane(
        __builtin_amdgcn_raw_buffer_load_b32(rme, 0, CNT_OFF + b * 128, AUX_SYS));
    const unsigned call = cnt + 1u;
    const unsigned cur = cnt % 3u;
    const unsigned clr = (cnt + 2u) % 3u;

    if (pusher) {
        xv.x = xv.x ? xv.x : 0x8000u;
        xv.y = xv.y ? xv.y : 0x8000u;
        xv.z = xv.z ? xv.z : 0x8000u;
        xv.w = xv.w ? xv.w : 0x8000u;
        const __amdgpu_buffer_rsrc_t rd = rsrc(a.ws[dest], WS_USED);
        __builtin_amdgcn_raw_buffer_store_b128(xv, rd, lane * 16,
                                               cur * BUF_BYTES + rank * SRC_BYTES + frag * 1024, AUX_SYS);
    } else if (wave >= 4) {
        const u32x4 z = {0u, 0u, 0u, 0u};
        __builtin_amdgcn_raw_buffer_store_b128(z, rme, lane * 16,
                                               clr * BUF_BYTES + b * 4096 + (wave & 3) * 1024, AUX_SYS);
    }
    STAMP(1);

    u32x4 bf[8][4];
    {
        const int voff = (lane >> 4) * (8 * NTOT * 2) + (lane & 15) * 8;
        const int sbase = (int)src * (KLOC * NTOT * 2) + nt * 128;
        u32x2 raw[8][8];
#pragma unroll
        for (int ks = 0; ks < 8; ++ks)
#pragma unroll
            for (int j = 0; j < 8; ++j)
                raw[ks][j] = __builtin_amdgcn_raw_buffer_load_b64(rw, voff, sbase + (ks * 32 + j) * (NTOT * 2), 0);
#pragma unroll
        for (int ks = 0; ks < 8; ++ks)
#pragma unroll
            for (int jj = 0; jj < 4; ++jj) {
                const u32x2 r0 = raw[ks][2 * jj], r1 = raw[ks][2 * jj + 1];
                bf[ks][0][jj] = __builtin_amdgcn_perm(r1.x, r0.x, 0x05040100u);
                bf[ks][1][jj] = __builtin_amdgcn_perm(r1.x, r0.x, 0x07060302u);
                bf[ks][2][jj] = __builtin_amdgcn_perm(r1.y, r0.y, 0x05040100u);
                bf[ks][3][jj] = __builtin_amdgcn_perm(r1.y, r0.y, 0x07060302u);
            }
    }
    STAMP(2);

    f32x4 acc[2][4];
#pragma unroll
    for (int rb = 0; rb < 2; ++rb)
#pragma unroll
        for (int f = 0; f < 4; ++f) acc[rb][f] = f32x4{0.f, 0.f, 0.f, 0.f};

    bool bad = false;
    if (wave == 0) {
        const int voff = (lane & 15) * (KLOC * 2) + (lane >> 4) * 16;
        const int sbase = ((int)rank * MLOC + 32 * mt) * (KLOC * 2);
        u32x4 av[16];
#pragma unroll
        for (int f = 0; f < 16; ++f)
            av[f] = __builtin_amdgcn_raw_buffer_load_b128(rx, voff, sbase + (f >> 3) * 16 * (KLOC * 2) + (f & 7) * 64, 0);
#pragma unroll
        for (int f = 0; f < 16; ++f)
#pragma unroll
            for (int n = 0; n < 4; ++n)
                acc[f >> 3][n] = __builtin_amdgcn_mfma_f32_16x16x32_bf16(
                    __builtin_bit_cast(bf16x8, av[f]), __builtin_bit_cast(bf16x8, bf[f & 7][n]), acc[f >> 3][n], 0, 0, 0);
    } else {
        const int sbase = (int)(cur * BUF_BYTES + src * SRC_BYTES) + mt * 16384;
        unsigned pend = 0xFFFFu;
        const unsigned long long t0 = __builtin_amdgcn_s_memrealtime();
        u32x4 av[16];
        while (true) {
            asm volatile("" ::: "memory");
#pragma unroll
            for (int f = 0; f < 16; ++f)
                if (pend & (1u << f))
                    av[f] = __builtin_amdgcn_raw_buffer_load_b128(rme, lane * 16, sbase + f * 1024, AUX_SYS);
#pragma unroll
            for (int f = 0; f < 16; ++f)
                if (pend & (1u << f)) {
                    const unsigned m = min(min(av[f].x, av[f].y), min(av[f].z, av[f].w));
                    if (__builtin_amdgcn_ballot_w64(m == 0u) == 0ull) {
#pragma unroll
                        for (int n = 0; n < 4; ++n)
                            acc[f >> 3][n] = __builtin_amdgcn_mfma_f32_16x16x32_bf16(
                                __builtin_bit_cast(bf16x8, av[f]), __builtin_bit_cast(bf16x8, bf[f & 7][n]), acc[f >> 3][n], 0, 0, 0);
                        pend &= ~(1u << f);
                    }
                }
            if (pend == 0u) break;
            if (__builtin_amdgcn_s_memrealtime() - t0 > DEADLINE_TICKS) { bad = true; break; }
        }
    }
    STAMP(3);

    if (bad) {
        const float qnan = __builtin_nanf("");
        acc[0][0] = f32x4{qnan, qnan, qnan, qnan};
        acc[1][0] = acc[0][0];
        acc[0][1] = acc[0][0]; acc[0][2] = acc[0][0]; acc[0][3] = acc[0][0];
        acc[1][1] = acc[0][0]; acc[1][2] = acc[0][0]; acc[1][3] = acc[0][0];
        if (lane == 0) __builtin_amdgcn_raw_buffer_store_b32(call, rme, 0, TMO_OFF, AUX_SYS);
    }
    {
        float* dst = red + wave * 2048 + (lane >> 4) * 4 * 64 + (lane & 15) * 4;
#pragma unroll
        for (int rb = 0; rb < 2; ++rb)
#pragma unroll
            for (int r = 0; r < 4; ++r) {
                const f32x4 v = {acc[rb][0][r], acc[rb][1][r], acc[rb][2][r], acc[rb][3][r]};
                *(f32x4*)(dst + (rb * 16 + r) * 64) = v;
            }
    }
    __syncthreads();
    STAMP(4);
    {
        const float* s = red + wave * 256 + lane * 4;
        f32x4 v = *(const f32x4*)s;
#pragma unroll
        for (int w = 1; w < 8; ++w) v += *(const f32x4*)(s + w * 2048);
        const bf16x2 lo = {(__bf16)gelu(v.x), (__bf16)gelu(v.y)};
        const bf16x2 hi = {(__bf16)gelu(v.z), (__bf16)gelu(v.w)};
        const u32x2 o = {__builtin_bit_cast(unsigned, lo), __builtin_bit_cast(unsigned, hi)};
        const __amdgpu_buffer_rsrc_t ro = rsrc(a.out, OUT_BYTES);
        const int voff = (lane >> 4) * (NTOT * 2) + (lane & 15) * 8;
        const int soff = (32 * mt + 4 * wave) * (NTOT * 2) + nt * 128;
        __builtin_amdgcn_raw_buffer_store_b64(o, ro, voff, soff, 0);
    }
    if (tid == 0) __builtin_amdgcn_raw_buffer_store_b32(call, rme, 0, CNT_OFF + b * 128, AUX_SYS);
    STAMP(5);
}

}

extern "C" void kernel_launch(int rank, int nranks, ncclComm_t comm,
                              hipStream_t stream,
                              void* const* d_in, const int* in_sizes, int n_in,
                              void* d_out, int out_size,
                              const int* in_dtypes, int out_dtype,
                              void* const* d_out_all,
                              void* const* d_ws_all, size_t ws_bytes) {
    (void)comm; (void)in_dtypes; (void)out_dtype; (void)d_out_all;
    if (nranks != NR || (unsigned)rank >= (unsigned)NR || n_in < 2 || ws_bytes < WS_USED ||
        in_sizes[0] != 2048 * KLOC || in_sizes[1] != KTOT * NTOT || out_size != MLOC * NTOT) {
        fprintf(stderr, "unexpected problem shape\n");
        return;
    }
    Args a;
    a.x = d_in[0];
    a.w = d_in[1];
    a.out = d_out;
    for (int p = 0; p < NR; ++p) a.ws[p] = d_ws_all[p];
    a.rank = rank;
    hipLaunchKernelGGL(a2a_gemm_gelu, dim3(256), dim3(512), 0, stream, a);
    HIP_CHECK(hipGetLastError());
}
```

```cpp
#include <hip/hip_runtime.h>
#include <rccl/rccl.h>
#include <cstdio>
#include <cstdint>

#define HIP_CHECK(expr) do { hipError_t _e = (expr); if (_e != hipSuccess) { \
    fprintf(stderr, "HIP error %s at %s:%d\n", hipGetErrorString(_e), __FILE__, __LINE__); \
    return; } } while (0)

typedef __attribute__((ext_vector_type(8))) __bf16 bf16x8;
typedef __attribute__((ext_vector_type(2))) __bf16 bf16x2;
typedef __attribute__((ext_vector_type(4))) float f32x4;
typedef __attribute__((ext_vector_type(2))) float f32x2;
typedef __attribute__((ext_vector_type(4))) unsigned u32x4;
typedef __attribute__((ext_vector_type(2))) unsigned u32x2;

namespace {

constexpr int NR = 8;
constexpr int MLOC = 256;
constexpr int KLOC = 256;
constexpr int KTOT = 2048, NTOT = 2048;
constexpr unsigned X_BYTES = 2048u * KLOC * 2u;
constexpr unsigned W_BYTES = (unsigned)KTOT * NTOT * 2u;
constexpr unsigned OUT_BYTES = MLOC * NTOT * 2u;
constexpr unsigned BUF_BYTES = 1u << 20;
constexpr unsigned SRC_BYTES = 128u << 10;
constexpr unsigned CNT_OFF = 3u * BUF_BYTES;
constexpr unsigned TMO_OFF = CNT_OFF + 256u * 128u;
constexpr unsigned PROF_OFF = 4u * BUF_BYTES;
constexpr unsigned WS_USED = 5u * BUF_BYTES;
constexpr unsigned long long DEADLINE_TICKS = 100000000ull;
constexpr int AUX_SYS = 17;

struct Args {
    const void* x;
    const void* w;
    void* out;
    void* ws[NR];
    int rank;
    unsigned cs;
};

__device__ inline __amdgpu_buffer_rsrc_t rsrc(const void* p, unsigned bytes) {
    return __builtin_amdgcn_make_buffer_rsrc(const_cast<void*>(p), (short)0, (int)bytes, 0x00020000);
}

__device__ inline float gelu(float y) {
    const float k0 = -2.0f * 0.7978845608028654f * 1.4426950408889634f;
    const float k1 = k0 * 0.044715f;
    float t = __builtin_fmaf(y * y, k1, k0);
    float e = __builtin_amdgcn_exp2f(y * t);
    return y * __builtin_amdgcn_rcpf(1.0f + e);
}

#define PTAB 0x43210000u
#define PTAB_SLOW 0x00000000u
#define STAB 0x32107654u
#define AHEAD 3
#define A_AUX 17

#define STAMP(i) do {} while (0)

#define WAIT_VM(n) __builtin_amdgcn_s_waitcnt(0x0F70 | ((n) & 15) | (((n) >> 4) << 14))

__global__ __launch_bounds__(512, 2) void a2a_gemm_gelu(Args a) {
    __shared__ __attribute__((aligned(16))) float red[8 * 32 * 64 + 4];
    STAMP(0);
    const int tid = threadIdx.x;
    const int lane = tid & 63;
    const unsigned wave = __builtin_amdgcn_readfirstlane(tid >> 6);
    const unsigned b = blockIdx.x;
    const unsigned rank = (unsigned)a.rank & 7u;
    const unsigned mt = b >> 5;
    const unsigned nt = ((b & 7u) << 2) | ((b >> 3) & 3u);
    const unsigned src = (rank + wave) & 7u;

    const __amdgpu_buffer_rsrc_t rx = rsrc(a.x, X_BYTES);
    const __amdgpu_buffer_rsrc_t rw = rsrc(a.w, W_BYTES);
    const __amdgpu_buffer_rsrc_t rme = rsrc(a.ws[rank], WS_USED);

    const unsigned dsel = (a.cs + nt) & 7u;
    const unsigned dest = (rank + 1u + dsel) & 7u;
    const unsigned slot = dsel == 7u ? 15u : (STAB >> (4u * mt)) & 15u;
    const unsigned prb = 4u * (nt >> 3) + (wave & 2u) + (slot & 1u);
    const unsigned pks = (((slot & 6u) ^ (7u - dsel)) & 6u) | (wave & 1u);
    const int voffx = (lane & 15) * (KLOC * 2) + (lane >> 4) * 16;
    u32x4 xv = {0u, 0u, 0u, 0u};
    if (wave < 4u && dsel != 7u)
        xv = __builtin_amdgcn_raw_buffer_load_b128(rx, voffx, (dest * MLOC + 16u * prb) * (KLOC * 2) + pks * 64u, 0);
    const unsigned cntv = __builtin_amdgcn_raw_buffer_load_b32(rme, 0, CNT_OFF + b * 128u, AUX_SYS);
    unsigned pf = 0;
    if (wave == 7u)
        pf = __builtin_amdgcn_raw_buffer_load_b32(rx, (lane & 31) * 128, (rank * MLOC + 32u * mt + 8u * (nt & 3u)) * (KLOC * 2), 0);

    unsigned ord;
    {
        const unsigned p4 = 4u * (((dsel == 7u ? PTAB_SLOW : PTAB) >> (4u * mt)) & 7u), m4 = 4u * mt;
        const unsigned base = 0x76543210u;
        const unsigned lowm = (1u << p4) - 1u;
        const unsigned below = (1u << m4) - 1u;
        const unsigned upto = (below << 4) | 15u;
        ord = (base & lowm) | (mt << p4) | ((base & below & ~lowm) << 4) | (base & ~upto);
        ord ^= wave * 0x11111111u;
    }
    const int voffw = (lane >> 3) * (4 * NTOT * 2) + (lane & 7) * 16 + (int)(src * (KLOC * NTOT * 2) + nt * 128u);

    const bool local = wave == 0u;
    const __amdgpu_buffer_rsrc_t ra = rsrc(local ? a.x : a.ws[rank], local ? X_BYTES : WS_USED);
    const int voffa = local ? voffx : lane * 16;
    const unsigned ash = local ? 6u : 10u;

    u32x4 raw[24][2];
    u32x4 bf[8][4];
    f32x4 acc[2][4];
#pragma unroll
    for (int rb = 0; rb < 2; ++rb)
#pragma unroll
        for (int f = 0; f < 4; ++f) acc[rb][f] = f32x4{0.f, 0.f, 0.f, 0.f};

    unsigned cnt = 0, abase = 0, pend = 0;
    STAMP(1);
#pragma unroll
    for (int s = -AHEAD; s < 24; ++s) {
        if (s >= 0 && s + AHEAD < 24) WAIT_VM(2 * (AHEAD - 1));
        __builtin_amdgcn_sched_barrier(0);
        if (s == 0) {
            STAMP(2);
            cnt = __builtin_amdgcn_readfirstlane(cntv);
            const unsigned cur = cnt % 3u;
            abase = local ? (rank * MLOC + 32u * mt) * (KLOC * 2) : cur * BUF_BYTES + src * SRC_BYTES + mt * 16384u;
            if (wave < 4u) {
                xv.x = xv.x ? xv.x : 0x8000u;
                xv.y = xv.y ? xv.y : 0x8000u;
                xv.z = xv.z ? xv.z : 0x8000u;
                xv.w = xv.w ? xv.w : 0x8000u;
            }
        }
        if (s >= 0 && s < 16 && (s & 1) == 0 && slot == (unsigned)(s >> 1)) {
            const unsigned bufi = wave < 4u ? cnt % 3u : (cnt + 2u) % 3u;
            const __amdgpu_buffer_rsrc_t rd = rsrc(a.ws[wave < 4u ? dest : rank], WS_USED);
            __builtin_amdgcn_raw_buffer_store_b128(xv, rd, lane * 16,
                bufi * BUF_BYTES + (wave < 4u ? rank : dest) * SRC_BYTES + (prb * 8u + pks) * 1024u, AUX_SYS);
            STAMP(7);
        }
        const int n = s + AHEAD;
        if (n < 16) {
            const unsigned ks = (ord >> (4 * (n >> 1))) & 7u;
            raw[n][0] = __builtin_amdgcn_raw_buffer_load_b128(rw, voffw + (2 * (n & 1)) * (NTOT * 2), ks * (32 * NTOT * 2), 0);
            raw[n][1] = __builtin_amdgcn_raw_buffer_load_b128(rw, voffw + (2 * (n & 1) + 1) * (NTOT * 2), ks * (32 * NTOT * 2), 0);
        } else if (n < 24) {
            const unsigned ks = (ord >> (4 * ((n - 15) & 7))) & 7u;
            raw[n][0] = __builtin_amdgcn_raw_buffer_load_b128(ra, voffa, abase + (ks << ash), A_AUX);
            raw[n][1] = __builtin_amdgcn_raw_buffer_load_b128(ra, voffa + 8192, abase + (ks << ash), A_AUX);
        }
        __builtin_amdgcn_sched_barrier(0);
        if (s >= 0 && s < 16) {
            const u32x4 r0 = raw[s][0], r1 = raw[s][1];
            unsigned P[8];
#pragma unroll
            for (int d = 0; d < 4; ++d) {
                P[2 * d] = __builtin_amdgcn_perm(r1[d], r0[d], 0x05040100u);
                P[2 * d + 1] = __builtin_amdgcn_perm(r1[d], r0[d], 0x07060302u);
            }
#pragma unroll
            for (int f = 0; f < 4; ++f) {
                bf[s >> 1][f][s & 1] = __builtin_amdgcn_update_dpp(P[f], P[4 + f], 0x128, 0xF, 0xC, false);
                bf[s >> 1][f][2 + (s & 1)] = __builtin_amdgcn_update_dpp(P[4 + f], P[f], 0x128, 0xF, 0x3, false);
            }
            if (s == 15) STAMP(3);
        } else if (s >= 16) {
            const int g = (s - 15) & 7;
            const u32x4 r0 = raw[s][0], r1 = raw[s][1];
            const unsigned m = min(min(min(r0.x, r0.y), min(r0.z, r0.w)), min(min(r1.x, r1.y), min(r1.z, r1.w)));
            if (local || __builtin_amdgcn_ballot_w64(m == 0u) == 0ull) {
#pragma unroll
                for (int f = 0; f < 4; ++f) {
                    acc[0][f] = __builtin_amdgcn_mfma_f32_16x16x32_bf16(
                        __builtin_bit_cast(bf16x8, r0), __builtin_bit_cast(bf16x8, bf[g][f]), acc[0][f], 0, 0, 0);
                    acc[1][f] = __builtin_amdgcn_mfma_f32_16x16x32_bf16(
                        __builtin_bit_cast(bf16x8, r1), __builtin_bit_cast(bf16x8, bf[g][f]), acc[1][f], 0, 0, 0);
                }
            } else {
                pend |= 1u << g;
            }
        }
        __builtin_amdgcn_sched_barrier(0);
    }
    asm volatile("" :: "v"(pf));
    STAMP(4);

    bool bad = false;
    if (pend) {
        const unsigned long long t0 = __builtin_amdgcn_s_memrealtime();
        while (true) {
            asm volatile("" ::: "memory");
            u32x4 av[8][2];
#pragma unroll
            for (int g = 0; g < 8; ++g)
                if (pend & (1u << g)) {
                    const unsigned ks = (ord >> (4 * g)) & 7u;
                    av[g][0] = __builtin_amdgcn_raw_buffer_load_b128(rme, lane * 16, abase + (ks << 10), AUX_SYS);
                    av[g][1] = __builtin_amdgcn_raw_buffer_load_b128(rme, lane * 16 + 8192, abase + (ks << 10), AUX_SYS);
                }
#pragma unroll
            for (int g = 0; g < 8; ++g)
                if (pend & (1u << g)) {
                    const u32x4 r0 = av[g][0], r1 = av[g][1];
                    const unsigned m = min(min(min(r0.x, r0.y), min(r0.z, r0.w)), min(min(r1.x, r1.y), min(r1.z, r1.w)));
                    if (__builtin_amdgcn_ballot_w64(m == 0u) == 0ull) {
#pragma unroll
                        for (int f = 0; f < 4; ++f) {
                            acc[0][f] = __builtin_amdgcn_mfma_f32_16x16x32_bf16(
                                __builtin_bit_cast(bf16x8, r0), __builtin_bit_cast(bf16x8, bf[g][f]), acc[0][f], 0, 0, 0);
                            acc[1][f] = __builtin_amdgcn_mfma_f32_16x16x32_bf16(
                                __builtin_bit_cast(bf16x8, r1), __builtin_bit_cast(bf16x8, bf[g][f]), acc[1][f], 0, 0, 0);
                        }
                        pend &= ~(1u << g);
                    }
                }
            if (pend == 0u) break;
            if (__builtin_amdgcn_s_memrealtime() - t0 > DEADLINE_TICKS) { bad = true; break; }
        }
    }
    STAMP(5);

    if (bad) {
        const float qnan = __builtin_nanf("");
#pragma unroll
        for (int rb = 0; rb < 2; ++rb)
#pragma unroll
            for (int f = 0; f < 4; ++f) acc[rb][f] = f32x4{qnan, qnan, qnan, qnan};
        if (lane == 0) __builtin_amdgcn_raw_buffer_store_b32(cnt + 1u, rme, 0, TMO_OFF, AUX_SYS);
    }
    {
        float* dst = red + wave * 2048 + (lane >> 4) * 4 * 64 + (lane & 7) * 8 + ((lane >> 3) & 1) * 4;
#pragma unroll
        for (int rb = 0; rb < 2; ++rb)
#pragma unroll
            for (int r = 0; r < 4; ++r) {
                const f32x4 v = {acc[rb][0][r], acc[rb][1][r], acc[rb][2][r], acc[rb][3][r]};
                *(f32x4*)(dst + (rb * 16 + r) * 64) = v;
            }
    }
    __syncthreads();
    STAMP(6);
    {
        const float* s = red + wave * 256 + lane * 4;
        f32x4 v = *(const f32x4*)s;
#pragma unroll
        for (int w = 1; w < 8; ++w) v += *(const f32x4*)(s + w * 2048);
        const bf16x2 lo = {(__bf16)gelu(v.x), (__bf16)gelu(v.y)};
        const bf16x2 hi = {(__bf16)gelu(v.z), (__bf16)gelu(v.w)};
        const u32x2 o = {__builtin_bit_cast(unsigned, lo), __builtin_bit_cast(unsigned, hi)};
        const __amdgpu_buffer_rsrc_t ro = rsrc(a.out, OUT_BYTES);
        const int voff = (lane >> 4) * (NTOT * 2) + (lane & 15) * 8;
        __builtin_amdgcn_raw_buffer_store_b64(o, ro, voff, (32u * mt + 4u * wave) * (NTOT * 2) + nt * 128u, 0);
    }
    if (tid == 0) __builtin_amdgcn_raw_buffer_store_b32(cnt + 1u, rme, 0, CNT_OFF + b * 128u, AUX_SYS);
}

}

extern "C" void kernel_launch(int rank, int nranks, ncclComm_t comm,
                              hipStream_t stream,
                              void* const* d_in, const int* in_sizes, int n_in,
                              void* d_out, int out_size,
                              const int* in_dtypes, int out_dtype,
                              void* const* d_out_all,
                              void* const* d_ws_all, size_t ws_bytes) {
    (void)comm; (void)in_dtypes; (void)out_dtype; (void)d_out_all;
    if (nranks != NR || (unsigned)rank >= (unsigned)NR || n_in < 2 || ws_bytes < WS_USED ||
        in_sizes[0] != 2048 * KLOC || in_sizes[1] != KTOT * NTOT || out_size != MLOC * NTOT) {
        fprintf(stderr, "unexpected problem shape\n");
        return;
    }
    Args a;
    a.x = d_in[0];
    a.w = d_in[1];
    a.out = d_out;
    for (int p = 0; p < NR; ++p) a.ws[p] = d_ws_all[p];
    a.rank = rank;
    a.cs = (unsigned)(((uintptr_t)d_in[1] >> 7) + 4u) & 7u;
    hipLaunchKernelGGL(a2a_gemm_gelu, dim3(256), dim3(512), 0, stream, a);
    HIP_CHECK(hipGetLastError());
}
```

```cpp
#include <hip/hip_runtime.h>
#include <rccl/rccl.h>
#include <cstdio>
#include <cstdint>

#define HIP_CHECK(expr) do { hipError_t _e = (expr); if (_e != hipSuccess) { \
    fprintf(stderr, "HIP error %s at %s:%d\n", hipGetErrorString(_e), __FILE__, __LINE__); \
    return; } } while (0)

typedef __attribute__((ext_vector_type(8))) __bf16 bf16x8;
typedef __attribute__((ext_vector_type(2))) __bf16 bf16x2;
typedef __attribute__((ext_vector_type(4))) float f32x4;
typedef __attribute__((ext_vector_type(2))) float f32x2;
typedef __attribute__((ext_vector_type(4))) unsigned u32x4;
typedef __attribute__((ext_vector_type(2))) unsigned u32x2;

namespace {

constexpr int NR = 8;
constexpr int MLOC = 256;
constexpr int KLOC = 256;
constexpr int KTOT = 2048, NTOT = 2048;
constexpr unsigned X_BYTES = 2048u * KLOC * 2u;
constexpr unsigned W_BYTES = (unsigned)KTOT * NTOT * 2u;
constexpr unsigned OUT_BYTES = MLOC * NTOT * 2u;
constexpr unsigned BUF_BYTES = 1u << 20;
constexpr unsigned SRC_BYTES = 128u << 10;
constexpr unsigned CNT_OFF = 3u * BUF_BYTES;
constexpr unsigned TMO_OFF = CNT_OFF + 256u * 128u;
constexpr unsigned PROF_OFF = 4u * BUF_BYTES;
constexpr unsigned WS_USED = 5u * BUF_BYTES;
constexpr unsigned long long DEADLINE_TICKS = 100000000ull;
constexpr int AUX_SYS = 17;

struct Args {
    const void* x;
    const void* w;
    void* out;
    void* ws[NR];
    int rank;
    unsigned cs;
};

__device__ inline __amdgpu_buffer_rsrc_t rsrc(const void* p, unsigned bytes) {
    return __builtin_amdgcn_make_buffer_rsrc(const_cast<void*>(p), (short)0, (int)bytes, 0x00020000);
}

__device__ inline float gelu(float y) {
    const float k0 = -2.0f * 0.7978845608028654f * 1.4426950408889634f;
    const float k1 = k0 * 0.044715f;
    float t = __builtin_fmaf(y * y, k1, k0);
    float e = __builtin_amdgcn_exp2f(y * t);
    return y * __builtin_amdgcn_rcpf(1.0f + e);
}

#define PTAB 0x43210000u
#define PTAB_SLOW 0x00000000u
#define STAB 0x32107654u
#define STEPT 0xECA86420u
#define AORD 0x76435210u
#define XNI 1
#define CLRMODE 1
#define AHEAD 3
#define A_AUX 17

#define STAMP(i) do {} while (0)

#define WAIT_VM(n) __builtin_amdgcn_s_waitcnt(0x0F70 | ((n) & 15) | (((n) >> 4) << 14))

__global__ __launch_bounds__(512, 2) void a2a_gemm_gelu(Args a) {
    __shared__ __attribute__((aligned(16))) float red[8 * 32 * 64 + 4];
    STAMP(0);
    const int tid = threadIdx.x;
    const int lane = tid & 63;
    const unsigned wave = __builtin_amdgcn_readfirstlane(tid >> 6);
    const unsigned b = blockIdx.x;
    const unsigned rank = (unsigned)a.rank & 7u;
    const unsigned mt = b >> 5;
    const unsigned nt = ((b & 7u) << 2) | ((b >> 3) & 3u);
    const unsigned src = (rank + wave) & 7u;

    const __amdgpu_buffer_rsrc_t rx = rsrc(a.x, X_BYTES);
    const __amdgpu_buffer_rsrc_t rw = rsrc(a.w, W_BYTES);
    const __amdgpu_buffer_rsrc_t rme = rsrc(a.ws[rank], WS_USED);

    const unsigned dsel = (a.cs + nt) & 7u;
    const unsigned dest = (rank + 1u + dsel) & 7u;
    const unsigned slot = dsel == 7u ? 15u : (STAB >> (4u * mt)) & 15u;
    const unsigned prb = 4u * (nt >> 3) + (wave & 2u) + (slot & 1u);
    const unsigned pks = (((slot & 6u) ^ (7u - dsel)) & 6u) | (wave & 1u);
    const int voffx = (lane & 15) * (KLOC * 2) + (lane >> 4) * 16;
    u32x4 xv = {0u, 0u, 0u, 0u};
    if (wave < 4u && dsel != 7u)
        xv = __builtin_amdgcn_raw_buffer_load_b128(rx, voffx, (dest * MLOC + 16u * prb) * (KLOC * 2) + pks * 64u, 0);
    const unsigned cntv = __builtin_amdgcn_raw_buffer_load_b32(rme, 0, CNT_OFF + b * 128u, AUX_SYS);
    unsigned pf = 0;
    if (wave == 7u)
        pf = __builtin_amdgcn_raw_buffer_load_b32(rx, (lane & 31) * 128, (rank * MLOC + 32u * mt + 8u * (nt & 3u)) * (KLOC * 2), 0);

    unsigned ord;
    {
        const unsigned p4 = 4u * (((dsel == 7u ? PTAB_SLOW : PTAB) >> (4u * mt)) & 7u), m4 = 4u * mt;
        const unsigned base = 0x76543210u;
        const unsigned lowm = (1u << p4) - 1u;
        const unsigned below = (1u << m4) - 1u;
        const unsigned upto = (below << 4) | 15u;
        ord = (base & lowm) | (mt << p4) | ((base & below & ~lowm) << 4) | (base & ~upto);
        ord ^= (wave ^ (XNI ? (nt & 3u) : 0u)) * 0x11111111u;
    }
    const unsigned sstep = slot < 8u ? (STEPT >> (4u * slot)) & 15u : 31u;
    const int voffw = (lane >> 3) * (4 * NTOT * 2) + (lane & 7) * 16 + (int)(src * (KLOC * NTOT * 2) + nt * 128u);

    const bool local = wave == 0u;
    const __amdgpu_buffer_rsrc_t ra = rsrc(local ? a.x : a.ws[rank], local ? X_BYTES : WS_USED);
    const int voffa = local ? voffx : lane * 16;
    const unsigned ash = local ? 6u : 10u;

    u32x4 raw[24][2];
    u32x4 bf[8][4];
    f32x4 acc[2][4];
#pragma unroll
    for (int rb = 0; rb < 2; ++rb)
#pragma unroll
        for (int f = 0; f < 4; ++f) acc[rb][f] = f32x4{0.f, 0.f, 0.f, 0.f};

    unsigned cnt = 0, abase = 0, pend = 0;
    STAMP(1);
#pragma unroll
    for (int s = -AHEAD; s < 24; ++s) {
        if (s >= 0 && s + AHEAD < 24) WAIT_VM(2 * (AHEAD - 1));
        __builtin_amdgcn_sched_barrier(0);
        if (s == 0) {
            STAMP(2);
            cnt = __builtin_amdgcn_readfirstlane(cntv);
            const unsigned cur = cnt % 3u;
            abase = local ? (rank * MLOC + 32u * mt) * (KLOC * 2) : cur * BUF_BYTES + src * SRC_BYTES + mt * 16384u;
            if (wave < 4u) {
                xv.x = xv.x ? xv.x : 0x8000u;
                xv.y = xv.y ? xv.y : 0x8000u;
                xv.z = xv.z ? xv.z : 0x8000u;
                xv.w = xv.w ? xv.w : 0x8000u;
            }
        }
        if (s >= 0 && s < 16 && sstep == (unsigned)s && (CLRMODE == 0 || wave < 4u)) {
            const unsigned bufi = wave < 4u ? cnt % 3u : (cnt + 2u) % 3u;
            const __amdgpu_buffer_rsrc_t rd = rsrc(a.ws[wave < 4u ? dest : rank], WS_USED);
            __builtin_amdgcn_raw_buffer_store_b128(xv, rd, lane * 16,
                bufi * BUF_BYTES + (wave < 4u ? rank : dest) * SRC_BYTES + (prb * 8u + pks) * 1024u, AUX_SYS);
            if (CLRMODE == 1)
                __builtin_amdgcn_raw_buffer_store_b128(u32x4{0u, 0u, 0u, 0u}, rme, lane * 16,
                    ((cnt + 2u) % 3u) * BUF_BYTES + dest * SRC_BYTES + (prb * 8u + pks) * 1024u, AUX_SYS);
            STAMP(7);
        }
        const int n = s + AHEAD;
        if (n < 16) {
            const unsigned ks = (ord >> (4 * (n >> 1))) & 7u;
            raw[n][0] = __builtin_amdgcn_raw_buffer_load_b128(rw, voffw + (2 * (n & 1)) * (NTOT * 2), ks * (32 * NTOT * 2), 0);
            raw[n][1] = __builtin_amdgcn_raw_buffer_load_b128(rw, voffw + (2 * (n & 1) + 1) * (NTOT * 2), ks * (32 * NTOT * 2), 0);
        } else if (n < 24) {
            const unsigned ks = (ord >> (4 * ((AORD >> (4 * (n - 16))) & 7u))) & 7u;
            raw[n][0] = __builtin_amdgcn_raw_buffer_load_b128(ra, voffa, abase + (ks << ash), A_AUX);
            raw[n][1] = __builtin_amdgcn_raw_buffer_load_b128(ra, voffa + 8192, abase + (ks << ash), A_AUX);
        }
        __builtin_amdgcn_sched_barrier(0);
        if (s >= 0 && s < 16) {
            const u32x4 r0 = raw[s][0], r1 = raw[s][1];
            unsigned P[8];
#pragma unroll
            for (int d = 0; d < 4; ++d) {
                P[2 * d] = __builtin_amdgcn_perm(r1[d], r0[d], 0x05040100u);
                P[2 * d + 1] = __builtin_amdgcn_perm(r1[d], r0[d], 0x07060302u);
            }
#pragma unroll
            for (int f = 0; f < 4; ++f) {
                bf[s >> 1][f][s & 1] = __builtin_amdgcn_update_dpp(P[f], P[4 + f], 0x128, 0xF, 0xC, false);
                bf[s >> 1][f][2 + (s & 1)] = __builtin_amdgcn_update_dpp(P[4 + f], P[f], 0x128, 0xF, 0x3, false);
            }
            if (s == 15) STAMP(3);
        } else if (s >= 16) {
            const int g = (AORD >> (4 * (s - 16))) & 7u;
            const u32x4 r0 = raw[s][0], r1 = raw[s][1];
            const unsigned m = min(min(min(r0.x, r0.y), min(r0.z, r0.w)), min(min(r1.x, r1.y), min(r1.z, r1.w)));
            if (local || __builtin_amdgcn_ballot_w64(m == 0u) == 0ull) {
#pragma unroll
                for (int f = 0; f < 4; ++f) {
                    acc[0][f] = __builtin_amdgcn_mfma_f32_16x16x32_bf16(
                        __builtin_bit_cast(bf16x8, r0), __builtin_bit_cast(bf16x8, bf[g][f]), acc[0][f], 0, 0, 0);
                    acc[1][f] = __builtin_amdgcn_mfma_f32_16x16x32_bf16(
                        __builtin_bit_cast(bf16x8, r1), __builtin_bit_cast(bf16x8, bf[g][f]), acc[1][f], 0, 0, 0);
                }
            } else {
                pend |= 1u << g;
            }
        }
        __builtin_amdgcn_sched_barrier(0);
    }
    asm volatile("" :: "v"(pf));
    STAMP(4);

    bool bad = false;
    if (pend) {
        const unsigned long long t0 = __builtin_amdgcn_s_memrealtime();
        while (true) {
            asm volatile("" ::: "memory");
            u32x4 av[8][2];
#pragma unroll
            for (int g = 0; g < 8; ++g)
                if (pend & (1u << g)) {
                    const unsigned ks = (ord >> (4 * g)) & 7u;
                    av[g][0] = __builtin_amdgcn_raw_buffer_load_b128(rme, lane * 16, abase + (ks << 10), AUX_SYS);
                    av[g][1] = __builtin_amdgcn_raw_buffer_load_b128(rme, lane * 16 + 8192, abase + (ks << 10), AUX_SYS);
                }
#pragma unroll
            for (int g = 0; g < 8; ++g)
                if (pend & (1u << g)) {
                    const u32x4 r0 = av[g][0], r1 = av[g][1];
                    const unsigned m = min(min(min(r0.x, r0.y), min(r0.z, r0.w)), min(min(r1.x, r1.y), min(r1.z, r1.w)));
                    if (__builtin_amdgcn_ballot_w64(m == 0u) == 0ull) {
#pragma unroll
                        for (int f = 0; f < 4; ++f) {
                            acc[0][f] = __builtin_amdgcn_mfma_f32_16x16x32_bf16(
                                __builtin_bit_cast(bf16x8, r0), __builtin_bit_cast(bf16x8, bf[g][f]), acc[0][f], 0, 0, 0);
                            acc[1][f] = __builtin_amdgcn_mfma_f32_16x16x32_bf16(
                                __builtin_bit_cast(bf16x8, r1), __builtin_bit_cast(bf16x8, bf[g][f]), acc[1][f], 0, 0, 0);
                        }
                        pend &= ~(1u << g);
                    }
                }
            if (pend == 0u) break;
            if (__builtin_amdgcn_s_memrealtime() - t0 > DEADLINE_TICKS) { bad = true; break; }
        }
    }
    STAMP(5);

    if (bad) {
        const float qnan = __builtin_nanf("");
#pragma unroll
        for (int rb = 0; rb < 2; ++rb)
#pragma unroll
            for (int f = 0; f < 4; ++f) acc[rb][f] = f32x4{qnan, qnan, qnan, qnan};
        if (lane == 0) __builtin_amdgcn_raw_buffer_store_b32(cnt + 1u, rme, 0, TMO_OFF, AUX_SYS);
    }
    {
        float* dst = red + wave * 2048 + (lane >> 4) * 4 * 64 + (lane & 7) * 8 + ((lane >> 3) & 1) * 4;
#pragma unroll
        for (int rb = 0; rb < 2; ++rb)
#pragma unroll
            for (int r = 0; r < 4; ++r) {
                const f32x4 v = {acc[rb][0][r], acc[rb][1][r], acc[rb][2][r], acc[rb][3][r]};
                *(f32x4*)(dst + (rb * 16 + r) * 64) = v;
            }
    }
    __syncthreads();
    STAMP(6);
    {
        const float* s = red + wave * 256 + lane * 4;
        f32x4 v = *(const f32x4*)s;
#pragma unroll
        for (int w = 1; w < 8; ++w) v += *(const f32x4*)(s + w * 2048);
        const bf16x2 lo = {(__bf16)gelu(v.x), (__bf16)gelu(v.y)};
        const bf16x2 hi = {(__bf16)gelu(v.z), (__bf16)gelu(v.w)};
        const u32x2 o = {__builtin_bit_cast(unsigned, lo), __builtin_bit_cast(unsigned, hi)};
        const __amdgpu_buffer_rsrc_t ro = rsrc(a.out, OUT_BYTES);
        const int voff = (lane >> 4) * (NTOT * 2) + (lane & 15) * 8;
        __builtin_amdgcn_raw_buffer_store_b64(o, ro, voff, (32u * mt + 4u * wave) * (NTOT * 2) + nt * 128u, 0);
    }
    if (CLRMODE == 2 && wave < 4u && slot < 8u)
        __builtin_amdgcn_raw_buffer_store_b128(u32x4{0u, 0u, 0u, 0u}, rme, lane * 16,
            ((cnt + 2u) % 3u) * BUF_BYTES + dest * SRC_BYTES + (prb * 8u + pks) * 1024u, AUX_SYS);
    if (tid == 0) __builtin_amdgcn_raw_buffer_store_b32(cnt + 1u, rme, 0, CNT_OFF + b * 128u, AUX_SYS);
}

}

extern "C" void kernel_launch(int rank, int nranks, ncclComm_t comm,
                              hipStream_t stream,
                              void* const* d_in, const int* in_sizes, int n_in,
                              void* d_out, int out_size,
                              const int* in_dtypes, int out_dtype,
                              void* const* d_out_all,
                              void* const* d_ws_all, size_t ws_bytes) {
    (void)comm; (void)in_dtypes; (void)out_dtype; (void)d_out_all;
    if (nranks != NR || (unsigned)rank >= (unsigned)NR || n_in < 2 || ws_bytes < WS_USED ||
        in_sizes[0] != 2048 * KLOC || in_sizes[1] != KTOT * NTOT || out_size != MLOC * NTOT) {
        fprintf(stderr, "unexpected problem shape\n");
        return;
    }
    Args a;
    a.x = d_in[0];
    a.w = d_in[1];
    a.out = d_out;
    for (int p = 0; p < NR; ++p) a.ws[p] = d_ws_all[p];
    a.rank = rank;
    a.cs = (unsigned)(((uintptr_t)d_in[1] >> 7) + 4u) & 7u;
    hipLaunchKernelGGL(a2a_gemm_gelu, dim3(256), dim3(512), 0, stream, a);
    HIP_CHECK(hipGetLastError());
}
```
